```python
import numpy as np

kernel_name = "gpu_gemm_ar_m4096_k4096_n4096_e4m3e4m3_dqrelu"


ABSMAX_RTOL = 0.03

COMPUTE_DTYPE = "float32"
EPILOGUE = "dequant_relu"
IN_QUANT = None
OUT_QUANT = None
IN_FP8 = "e4m3e4m3"
IN_DTYPES = {"x": "e4m3", "w_mat": "e4m3",
             "scale_x": "float32", "scale_w": "float32"}
WORLD_SIZE = 8

def _snap_fp8(x, mant_bits, max_abs, min_normal):
    a = np.abs(x.astype(np.float64))
    _m, _e = np.frexp(a)
    q = np.where(a >= min_normal, np.ldexp(1.0, _e - 1 - mant_bits),
                 min_normal * 2.0 ** -mant_bits)
    snapped = np.minimum(np.round(a / q) * q, max_abs)
    return (np.sign(x) * snapped).astype(np.float32)

def setup_inputs():

    x = _snap_fp8(np.random.randn(4096, 4096).astype(np.float32), 3, 448.0, 2.0 ** -6)
    w_mat = _snap_fp8(np.random.randn(4096, 4096).astype(np.float32), 3, 448.0, 2.0 ** -6)
    scale_x = (0.02 * (1.0 + np.random.rand(1))).astype(np.float32)
    scale_w = (0.02 * (1.0 + np.random.rand(1))).astype(np.float32)
    return {"x": x, "w_mat": w_mat, "scale_x": scale_x, "scale_w": scale_w}

def reference(x, w_mat, scale_x, scale_w):
    acc = (x @ w_mat).astype(np.float32)
    y = (acc * (scale_x * scale_w)).astype(np.float32)
    return np.maximum(y, 0.0).astype(np.float32)
```

```cpp
#include <hip/hip_runtime.h>
#include <rccl/rccl.h>
#include <cstdio>
#include <cstdint>

#define HIP_CHECK(expr) do { hipError_t _e = (expr); if (_e != hipSuccess) { \
    fprintf(stderr, "HIP error %s at %s:%d\n", hipGetErrorString(_e), __FILE__, __LINE__); \
    return; } } while (0)

typedef int      v8i __attribute__((ext_vector_type(8)));
typedef int      v4i __attribute__((ext_vector_type(4)));
typedef float    v4f __attribute__((ext_vector_type(4)));
typedef unsigned v4u __attribute__((ext_vector_type(4)));
#define LAS __attribute__((address_space(3)))

constexpr int NR    = 8;
constexpr int MM    = 4096;
constexpr int NN    = 4096;
constexpr int KS    = 512;
constexpr int NSTEP = 32;
constexpr int NPUSH = 16;
constexpr int NRB   = 15;
constexpr int NCB   = 16;
constexpr int NGEMM = NRB * NCB;
constexpr int GRID  = NPUSH + NGEMM;

constexpr size_t WS_TMO   = 0;
constexpr size_t WS_DONE  = 512;
constexpr size_t WS_TICKET = 1024;
constexpr size_t WS_CTR   = 16 * 1024;
constexpr size_t WS_FLAGS = 64 * 1024;
constexpr size_t WS_PROF  = 128 * 1024;
constexpr size_t WS_DATA  = 1u << 20;
constexpr size_t PANEL    = 4096 * 16;
constexpr size_t OPER     = (size_t)NSTEP * NR * PANEL;
constexpr size_t BUFB     = 2 * OPER;
constexpr size_t WS_NEED  = WS_DATA + 2 * BUFB;

constexpr int A_PAN   = 288 * 16;
constexpr int B_PAN   = 256 * 16;
constexpr int A_STG   = NR * A_PAN;
constexpr int B_STG   = NR * B_PAN;
constexpr int STG     = A_STG + B_STG;
constexpr int LDS_MISC = 2 * STG;
constexpr int LDS_BYTES = LDS_MISC + 512;

#define LEAD 100
#define DEPTH 1
#define POLL_SLEEP 2
#define OUT_AUX 16
constexpr unsigned long long DEADLINE = 40000000ull;

struct Params {
    const unsigned char* x;
    const unsigned char* w;
    const float* sx;
    const float* sw;
    float* out;
    unsigned char* ws[NR];
    int rank;
};

__device__ __forceinline__ void ld_flag_async(unsigned& f, const unsigned* p) {
    asm volatile("global_load_dword %0, %1, off sc0 sc1" : "+v"(f) : "v"(p) : "memory");
}
__device__ __forceinline__ void ld_flag_async_masked(unsigned& f, const unsigned* p, int mask  ) {
    unsigned long long ex;
    asm volatile("s_mov_b64 %1, exec\n\ts_mov_b32 exec_lo, %3\n\ts_mov_b32 exec_hi, %3\n\t"
                 "global_load_dword %0, %2, off sc0 sc1\n\ts_mov_b64 exec, %1"
                 : "+v"(f), "=&s"(ex) : "v"(p), "s"(mask) : "memory");
}
__device__ __forceinline__ unsigned ld_flag(const unsigned* p) {
    unsigned f;
    asm volatile("global_load_dword %0, %1, off sc0 sc1\n\ts_waitcnt vmcnt(0)" : "=&v"(f) : "v"(p) : "memory");
    return f;
}

__device__ __forceinline__ int uni32(int v) {
    int r;
    asm volatile("s_nop 0\n\tv_readfirstlane_b32 %0, %1\n\ts_nop 4" : "=s"(r) : "v"(v));
    return r;
}
__device__ __forceinline__ unsigned long long uni64(unsigned long long v) {
    const unsigned lo = (unsigned)uni32((int)(unsigned)v), hi = (unsigned)uni32((int)(unsigned)(v >> 32));
    return ((unsigned long long)hi << 32) | lo;
}

#define STAMP(i) do {} while (0)

__device__ __forceinline__ unsigned next_call(const Params& P) {
    unsigned* c = (unsigned*)(P.ws[0] + WS_CTR + (size_t)blockIdx.x * 64);
    return __hip_atomic_load(c, __ATOMIC_RELAXED, __HIP_MEMORY_SCOPE_AGENT) + 1u;
}
__device__ __forceinline__ void commit_call(const Params& P, unsigned call) {
    unsigned* c = (unsigned*)(P.ws[0] + WS_CTR + (size_t)blockIdx.x * 64);
    __hip_atomic_store(c, call, __ATOMIC_RELAXED, __HIP_MEMORY_SCOPE_AGENT);
}

__device__ __forceinline__ v4u w_transpose(unsigned w0, unsigned w1, unsigned w2, unsigned w3, int b) {
    const unsigned l01 = __builtin_amdgcn_perm(w1, w0, 0x05010400u), h01 = __builtin_amdgcn_perm(w1, w0, 0x07030602u);
    const unsigned l23 = __builtin_amdgcn_perm(w3, w2, 0x05010400u), h23 = __builtin_amdgcn_perm(w3, w2, 0x07030602u);
    const unsigned o0 = __builtin_amdgcn_perm(l23, l01, 0x05040100u), o1 = __builtin_amdgcn_perm(l23, l01, 0x07060302u);
    const unsigned o2 = __builtin_amdgcn_perm(h23, h01, 0x05040100u), o3 = __builtin_amdgcn_perm(h23, h01, 0x07060302u);
#define QB(x, j) ((unsigned)__builtin_amdgcn_update_dpp(0, (int)(x), (j) * 0x55, 0xf, 0xf, true))
    const unsigned t00 = QB(o0, 0), t01 = QB(o1, 0), t02 = QB(o2, 0), t03 = QB(o3, 0);
    const unsigned t10 = QB(o0, 1), t11 = QB(o1, 1), t12 = QB(o2, 1), t13 = QB(o3, 1);
    const unsigned t20 = QB(o0, 2), t21 = QB(o1, 2), t22 = QB(o2, 2), t23 = QB(o3, 2);
    const unsigned t30 = QB(o0, 3), t31 = QB(o1, 3), t32 = QB(o2, 3), t33 = QB(o3, 3);
    v4u r;
    r.x = b == 0 ? t00 : b == 1 ? t01 : b == 2 ? t02 : t03;
    r.y = b == 0 ? t10 : b == 1 ? t11 : b == 2 ? t12 : t13;
    r.z = b == 0 ? t20 : b == 1 ? t21 : b == 2 ? t22 : t23;
    r.w = b == 0 ? t30 : b == 1 ? t31 : b == 2 ? t32 : t33;
    return r;
}
__device__ __forceinline__ unsigned w_pos(int tid) {
    return (unsigned)(((tid >> 8) * 256 + (tid & 3) * 64 + ((tid >> 2) & 63)) * 16);
}

constexpr int NRELAY = 128;
__device__ __forceinline__ void relayout(const Params& P, int g, unsigned call) {
    const int tid = threadIdx.x;
    const int isw = g & 1, j = (g >> 1) & 7, rho = g >> 4;
    const __amdgpu_buffer_rsrc_t r0 = __builtin_amdgcn_make_buffer_rsrc(P.ws[0], 0, (int)WS_NEED, 0x00020000);
    const unsigned base = (unsigned)(WS_DATA + (size_t)(call & 1) * BUFB + (size_t)P.rank * PANEL
                                     + (size_t)rho * 4 * NR * PANEL + (size_t)j * 8192);
    if (!isw) {
        v4u v[4];
#pragma unroll
        for (int p = 0; p < 4; ++p)
            v[p] = *(const v4u*)(P.x + (size_t)(j * 512 + p * 128 + (tid >> 2)) * KS + rho * 64 + (tid & 3) * 16);
#pragma unroll
        for (int p = 0; p < 4; ++p)
            __builtin_amdgcn_raw_buffer_store_b128(v[p], r0, base + (unsigned)(tid & 3) * (unsigned)(NR * PANEL) + (unsigned)(p * 128 + (tid >> 2)) * 16u, 0, 17);
    } else {
        unsigned d[4][4];
#pragma unroll
        for (int p = 0; p < 4; ++p)
#pragma unroll
            for (int k = 0; k < 4; ++k)
                d[p][k] = *(const unsigned*)(P.w + (size_t)(rho * 64 + p * 16 + (tid & 3) * 4 + k) * NN + j * 512 + (tid >> 2) * 4);
#pragma unroll
        for (int p = 0; p < 4; ++p)
            __builtin_amdgcn_raw_buffer_store_b128(w_transpose(d[p][0], d[p][1], d[p][2], d[p][3], tid & 3), r0,
                                                   base + (unsigned)OPER + (unsigned)p * (unsigned)(NR * PANEL) + w_pos(tid), 0, 17);
    }
    asm volatile("s_waitcnt vmcnt(0)" ::: "memory");
    __builtin_amdgcn_s_barrier();
    asm volatile("" ::: "memory");
    if (tid < 4)
        __hip_atomic_store((unsigned*)(P.ws[0] + WS_FLAGS + (size_t)(rho * 4 + tid) * (NR * 128) + (size_t)P.rank * 128 + (isw * 8 + j) * 4),
                           call, __ATOMIC_RELAXED, __HIP_MEMORY_SCOPE_SYSTEM);
    if (tid == 64)
        __hip_atomic_fetch_add((unsigned*)(P.ws[0] + WS_DONE), 1u, __ATOMIC_RELAXED, __HIP_MEMORY_SCOPE_AGENT);
}

struct Raw { v4u x; unsigned w0, w1, w2, w3; };
constexpr int NDIRECT = 64;

__device__ __forceinline__ void pusher(const Params& P, LAS unsigned char* lds) {
    const int tid  = threadIdx.x;
    const int lane = tid & 63;
    const int wave = __builtin_amdgcn_readfirstlane(tid >> 6);
    const int pid = uni32((int)blockIdx.x);
    const int rank = P.rank;
    constexpr int NUNIT = NSTEP * NPUSH;
    static_assert(NUNIT == 0x200 && NDIRECT == 64, "asm below");
    LAS unsigned* tk_lds = (LAS unsigned*)(lds + LDS_MISC);

    const unsigned xoff = (unsigned)tid * KS, poff = (unsigned)tid * 16u;
    const unsigned woffA = (unsigned)(((tid & 3) * 4 + 1) * NN + (tid >> 2) * 4), woffB = woffA + 2 * NN;
    const unsigned long long xb = uni64((unsigned long long)P.x), wb = uni64((unsigned long long)P.w);
    auto unit_off = [&](int u) -> unsigned {
        return (unsigned)(u >> 4) * (unsigned)(NR * PANEL) + (unsigned)(u & 7) * 8192u + ((u & 8) ? (unsigned)OPER : 0u);
    };
    auto load_unit = [&](Raw& r, int u, unsigned long long pbase) {
        const int s = (u >> 4) & 3, j = u & 7;
        const unsigned long long xa = xb + (unsigned long long)(j * 512 * KS + s * 16);
        const unsigned long long wa = wb + (unsigned long long)(s * 16 * NN + j * 512);
        const unsigned long long pa = pbase + unit_off(u & (NUNIT - 1));
        asm volatile("s_cmpk_lt_i32 %12, 0x200\n\t"
                     "s_cbranch_scc0 3f\n\t"
                     "s_cmpk_lt_i32 %12, 64\n\t"
                     "s_cbranch_scc1 1f\n\t"
                     "global_load_dwordx4 %0, %8, %11 sc1\n\t"
                     "s_branch 3f\n"
                     "1:\n\t"
                     "s_bitcmp0_b32 %12, 3\n\t"
                     "s_cbranch_scc0 2f\n\t"
                     "global_load_dwordx4 %0, %5, %9\n\t"
                     "s_branch 3f\n"
                     "2:\n\t"
                     "global_load_dword %1, %6, %10 offset:-4096\n\t"
                     "global_load_dword %2, %6, %10\n\t"
                     "global_load_dword %3, %7, %10 offset:-4096\n\t"
                     "global_load_dword %4, %7, %10\n"
                     "3:"
                     : "+v"(r.x), "+v"(r.w0), "+v"(r.w1), "+v"(r.w2), "+v"(r.w3)
                     : "v"(xoff), "v"(woffA), "v"(woffB), "v"(poff), "s"(xa), "s"(wa), "s"(pa), "s"(u)
                     : "memory", "scc");
    };

    Raw ra = {{0u, 0u, 0u, 0u}, 0u, 0u, 0u, 0u}, rb = ra;
    int u0 = pid, u1 = NPUSH + pid, u2 = 2 * NPUSH + pid, u3 = 3 * NPUSH + pid, up = -1;
    load_unit(ra, u0, xb);
    load_unit(rb, u1, xb);
    const unsigned call = __builtin_amdgcn_readfirstlane(next_call(P));
    unsigned* tkc = (unsigned*)(P.ws[0] + WS_TICKET + (size_t)(call & 1) * 128);
    if (pid == 0 && tid == 0)
        __hip_atomic_store((unsigned*)(P.ws[0] + WS_TICKET + (size_t)((call + 1) & 1) * 128), 0u, __ATOMIC_RELAXED, __HIP_MEMORY_SCOPE_AGENT);

    __amdgpu_buffer_rsrc_t rs[NR];
#pragma unroll
    for (int dd = 1; dd < NR; ++dd) rs[dd] = __builtin_amdgcn_make_buffer_rsrc(P.ws[dd], 0, (int)WS_NEED, 0x00020000);
    unsigned char* fb = P.ws[1];
#pragma unroll
    for (int q = 2; q < NR; ++q) fb = ((lane & 7) == q) ? P.ws[q] : fb;
    fb += WS_FLAGS + (size_t)rank * 128;
    const unsigned dbase = (unsigned)(WS_DATA + (size_t)(call & 1) * BUFB + (size_t)rank * PANEL);
    const unsigned long long pbase = uni64((unsigned long long)(P.ws[0] + dbase));
    const unsigned wpos = w_pos(tid);
    auto publish = [&](int q) {
        if (wave == 1 + ((q >> 4) & 3) && lane >= 1 && lane < NR)
            __hip_atomic_store((unsigned*)(fb + (size_t)(q >> 4) * (NR * 128) + (q & 15) * 4), call, __ATOMIC_RELAXED, __HIP_MEMORY_SCOPE_SYSTEM);
    };

    unsigned tka = 0, tkb = 0;
    asm volatile("s_waitcnt vmcnt(0)" : "+v"(ra.x), "+v"(ra.w0), "+v"(ra.w1), "+v"(ra.w2), "+v"(ra.w3),
                                        "+v"(rb.x), "+v"(rb.w0), "+v"(rb.w1), "+v"(rb.w2), "+v"(rb.w3) :: "memory");
    int it = 0;
    auto step = [&](Raw& rc, Raw& ro, unsigned& tkn, unsigned& tko) {
        const bool dw = (u0 < NDIRECT) && (u0 & 8);
        v4u item = rc.x;
        if (dw) item = w_transpose(rc.w0, rc.w1, rc.w2, rc.w3, tid & 3);
        const unsigned off = dbase + unit_off(u0) + (dw ? wpos : poff);
#pragma unroll
        for (int dd = 1; dd < NR; ++dd)
            __builtin_amdgcn_raw_buffer_store_b128(item, rs[dd], off, 0, 17);
        if (it == 2) {
            const unsigned* dn = (const unsigned*)(P.ws[0] + WS_DONE);
            const unsigned long long t0 = __builtin_amdgcn_s_memrealtime();
            while (ld_flag(dn) - call * (unsigned)NRELAY > 0x7fffffffu) {
                if (__builtin_amdgcn_s_memrealtime() - t0 > DEADLINE) {
                    __hip_atomic_store((unsigned*)(P.ws[0] + WS_TMO), call, __ATOMIC_RELAXED, __HIP_MEMORY_SCOPE_AGENT);
                    break;
                }
                __builtin_amdgcn_s_sleep(1);
            }
        }
        load_unit(rc, u2, pbase);
        if (wave == 0) {
            unsigned long long ex;
            asm volatile("s_mov_b64 %1, exec\n\ts_mov_b64 exec, 1\n\t"
                         "global_atomic_add %0, %2, %3, off sc0\n\ts_mov_b64 exec, %1"
                         : "+v"(tkn), "=&s"(ex) : "v"(tkc), "v"(1u) : "memory");
        }
        if (u2 >= NUNIT)                     asm volatile("s_waitcnt vmcnt(7)" ::: "memory");
        else if (u2 < NDIRECT && (u2 & 8))   asm volatile("s_waitcnt vmcnt(11)" ::: "memory");
        else                                 asm volatile("s_waitcnt vmcnt(8)" ::: "memory");
        asm volatile("" : "+v"(tko), "+v"(ro.x), "+v"(ro.w0), "+v"(ro.w1), "+v"(ro.w2), "+v"(ro.w3));
        if (tid == 0) tk_lds[it & 1] = tko;
        asm volatile("s_waitcnt lgkmcnt(0)" ::: "memory");
        __builtin_amdgcn_s_barrier();
        asm volatile("" ::: "memory");
        if (it == 0 && tid == 0) commit_call(P, call);
        if (up >= 0) publish(up);
        up = u0; u0 = u1; u1 = u2; u2 = u3;
        if (it == 0) {
            u3 = 4 * NPUSH + pid;
        } else {
            const unsigned t = __builtin_amdgcn_readfirstlane(tk_lds[it & 1]);
            u3 = 5 * NPUSH + (int)(t < 4096u ? t : 4096u);
        }
        ++it;
    };
    while (true) {
        step(ra, rb, tka, tkb);
        if (u0 >= NUNIT) break;
        step(rb, ra, tkb, tka);
        if (u0 >= NUNIT) break;
    }
    asm volatile("s_waitcnt vmcnt(0)" : "+v"(tka), "+v"(tkb) :: "memory");
    __builtin_amdgcn_s_barrier();
    asm volatile("" ::: "memory");
    publish(up);
}

#define MFMA(a, b, c) __builtin_amdgcn_mfma_scale_f32_16x16x128_f8f6f4((a), (b), (c), 0, 0, 0, 0, 0, 0)

__device__ __forceinline__ v8i ld_frag(const LAS unsigned char* p, int pan) {
    const v4i lo = *(const LAS v4i*)(p);
    const v4i hi = *(const LAS v4i*)(p + pan);
    return __builtin_shufflevector(lo, hi, 0, 1, 2, 3, 4, 5, 6, 7);
}

__device__ __forceinline__ void gemm(const Params& P, LAS unsigned char* lds) {
    const unsigned call = __builtin_amdgcn_readfirstlane(next_call(P));
    const int tid  = threadIdx.x;
    const int lane = tid & 63;
    const int wave = __builtin_amdgcn_readfirstlane(tid >> 6);
    const int wm = wave >> 2, wn = wave & 3;
    const int g   = blockIdx.x - NPUSH;
    if (g < NRELAY) relayout(P, g, call);
    const int xcd = g & 7, idx = g >> 3;
    const int cb = 2 * xcd + (idx & 1), rb = idx >> 1;
    const int row0 = rb * 272, col0 = cb * 256;
    const int nunits = (rb == NRB - 1) ? 18 : 17;
    const bool has9 = (wm == 0) || (nunits == 18);
    unsigned char* ws = P.ws[0];
    unsigned* tmo = (unsigned*)(ws + WS_TMO);

    const int fj = (lane == 0) ? (row0 >> 9) : (lane == 1) ? ((row0 + nunits * 16 - 1) >> 9) : (8 + (col0 >> 9));
    const unsigned* fptr = (const unsigned*)(ws + WS_FLAGS + wave * 128 + fj * 4);

    const unsigned long long ga = uni64((unsigned long long)(ws + WS_DATA + (size_t)(call & 1) * BUFB + (size_t)wave * PANEL + (size_t)row0 * 16 + 2048));
    const unsigned long long gb = uni64((unsigned long long)(ws + WS_DATA + (size_t)(call & 1) * BUFB + OPER + (size_t)wave * PANEL + (size_t)col0 * 16));
    const unsigned voff = lane * 16;
    const unsigned long long m5 = uni64((nunits == 18) ? 0xffffffffull : 0xffffull);
    const unsigned lbase = (unsigned)(size_t)lds;
    const unsigned la = (unsigned)uni32((int)(lbase + wave * A_PAN + 2048));
    const unsigned lb = (unsigned)uni32((int)(lbase + A_STG + wave * B_PAN));

    auto issue = [&](int s) {
        const unsigned long long pa = ga + (unsigned long long)s * (NR * PANEL);
        const unsigned long long pb = gb + (unsigned long long)s * (NR * PANEL);
        const unsigned bo = (s & 1) * STG;
        unsigned keep; unsigned long long ex;
        asm volatile(
            "s_mov_b32 %0, m0\n\t"
            "s_mov_b32 m0, %5\n\t"
            "s_nop 0\n\t"
            "global_load_lds_dwordx4 %2, %4\n\t"
            "global_load_lds_dwordx4 %2, %4 offset:1024\n\t"
            "global_load_lds_dwordx4 %2, %4 offset:2048\n\t"
            "global_load_lds_dwordx4 %2, %4 offset:3072\n\t"
            "s_mov_b32 m0, %6\n\t"
            "s_mov_b64 %1, exec\n\t"
            "global_load_lds_dwordx4 %2, %3 offset:-2048\n\t"
            "global_load_lds_dwordx4 %2, %3 offset:-1024\n\t"
            "global_load_lds_dwordx4 %2, %3\n\t"
            "global_load_lds_dwordx4 %2, %3 offset:1024\n\t"
            "s_mov_b64 exec, %7\n\t"
            "global_load_lds_dwordx4 %2, %3 offset:2048\n\t"
            "s_mov_b64 exec, %1\n\t"
            "s_mov_b32 m0, %0"
            : "=&s"(keep), "=&s"(ex)
            : "v"(voff), "s"(pa), "s"(pb), "s"(lb + bo), "s"(la + bo), "s"(m5)
            : "memory");
    };
    bool dead = false;
    auto wait_flags = [&](int s, unsigned f) {
        if (__all(f >= call) || dead) return;
        const unsigned long long t0 = __builtin_amdgcn_s_memrealtime();
        for (unsigned it = 1;; ++it) {
            __builtin_amdgcn_s_sleep(POLL_SLEEP);
            f = ld_flag(fptr + s * (NR * 32));
            if (__all(f >= call)) return;
            if ((it & 63u) == 0u) {
                if (__hip_atomic_load(tmo, __ATOMIC_RELAXED, __HIP_MEMORY_SCOPE_AGENT) == call
                    || __builtin_amdgcn_s_memrealtime() - t0 > DEADLINE) {
                    __hip_atomic_store(tmo, call, __ATOMIC_RELAXED, __HIP_MEMORY_SCOPE_AGENT);
                    dead = true;
                    return;
                }
            }
        }
    };

    v4f acc[9][4];
#pragma unroll
    for (int m = 0; m < 9; ++m)
#pragma unroll
        for (int b = 0; b < 4; ++b) acc[m][b] = v4f{0.f, 0.f, 0.f, 0.f};

    const int fr = lane & 15, fg = lane >> 4;
    const LAS unsigned char* pa[2];
    const LAS unsigned char* pb[2];
    pa[0] = lds + (2 * fg) * A_PAN + (wm * 144 + fr) * 16;
    pb[0] = lds + A_STG + (2 * fg) * B_PAN + (wn * 16 + fr) * 16;
    pa[1] = pa[0] + STG;
    pb[1] = pb[0] + STG;
    asm volatile("" : "+v"(pa[0]), "+v"(pb[0]), "+v"(pa[1]), "+v"(pb[1]));

    wait_flags(0, ld_flag(fptr));
    issue(0);
    wait_flags(1, ld_flag(fptr + NR * 32));
    issue(1);
    asm volatile("s_waitcnt vmcnt(9)" ::: "memory");
    __builtin_amdgcn_s_barrier();
    asm volatile("" ::: "memory");
    if (tid == 0) commit_call(P, call);

    v8i B[4], A[3];
    unsigned f = 0;
#pragma unroll
    for (int b = 0; b < 4; ++b) B[b] = ld_frag(pb[0] + b * 1024, B_PAN);
    A[0] = ld_frag(pa[0], A_PAN);
    A[1] = ld_frag(pa[0] + 256, A_PAN);

    int nxt = 2;
    int fs = -1;
    for (int s2 = 0; s2 < NSTEP; s2 += 2) {
#pragma unroll
        for (int h = 0; h < 2; ++h) {
            const int s = s2 + h;
            const bool more1 = (h == 0) || (s + 1 < NSTEP);
#pragma unroll
            for (int m = 0; m < 7; ++m) {
                if (m + 2 < 8 || has9) A[(m + 2) % 3] = ld_frag(pa[h] + (m + 2) * 256, A_PAN);
                if (m == 3) {
                    if (more1 && nxt == s + 1) asm volatile("s_waitcnt vmcnt(0)" ::: "memory");
                    asm volatile("" : "+v"(f));
                    if (more1 && nxt == s + 1 && __all(f >= call)) { issue(s + 1); nxt = s + 2; }
                    fs = nxt < NSTEP ? nxt : NSTEP - 1;
                    ld_flag_async(f, fptr + fs * (NR * 32));
                }
#pragma unroll
                for (int b = 0; b < 4; ++b) acc[m][b] = MFMA(A[m % 3], B[b], acc[m][b]);
            }
            __builtin_amdgcn_sched_barrier(0);
            if (more1 && nxt == s + 1) asm volatile("s_waitcnt vmcnt(0)" ::: "memory");
            asm volatile("" : "+v"(f));
            if (more1 && nxt == s + 1) {
                wait_flags(s + 1, f);
                issue(s + 1);
                nxt = s + 2;
            }
            asm volatile("s_waitcnt vmcnt(0) lgkmcnt(0)" : "+v"(f) :: "memory");
            __builtin_amdgcn_s_barrier();
            asm volatile("" ::: "memory");
            __builtin_amdgcn_sched_barrier(0);
            STAMP(2 + s);
            {
                const bool open = (nxt == s + 2) && (nxt < NSTEP);
                const bool ok = open && (fs == nxt) && __all(f >= call);
                ld_flag_async_masked(f, fptr + (open ? nxt : 0) * (NR * 32),
                                     __builtin_amdgcn_readfirstlane((open && !ok) ? -1 : 0));
                if (ok) { issue(s + 2); nxt = s + 3; }
            }
            v8i An;
            if (more1) An = ld_frag(pa[h ^ 1], A_PAN);
            if (has9) {
#pragma unroll
                for (int b = 0; b < 4; ++b) acc[8][b] = MFMA(A[2], B[b], acc[8][b]);
            }
#pragma unroll
            for (int b = 0; b < 4; ++b) {
                acc[7][b] = MFMA(A[1], B[b], acc[7][b]);
                if (more1) B[b] = ld_frag(pb[h ^ 1] + b * 1024, B_PAN);
            }
            if (more1) {
                A[0] = An;
                A[1] = ld_frag(pa[h ^ 1] + 256, A_PAN);
            }
        }
    }

    STAMP(40);
    const float sc = P.sx[0] * P.sw[0];
    const bool bad = __hip_atomic_load(tmo, __ATOMIC_RELAXED, __HIP_MEMORY_SCOPE_AGENT) == call;
    const __amdgpu_buffer_rsrc_t ro = __builtin_amdgcn_make_buffer_rsrc(P.out, 0, MM * NN * 4, 0x00020000);
    const unsigned ob = (unsigned)((row0 + wm * 144 + fg * 4) * NN + col0 + wn * 64 + fr * 4) * 4u;
#pragma unroll
    for (int m = 0; m < 9; ++m) {
        if (m < 8 || has9) {
#pragma unroll
            for (int j = 0; j < 4; ++j) {
                v4u v;
#pragma unroll
                for (int b = 0; b < 4; ++b) v[b] = bad ? 0x7fc00000u : __float_as_uint(fmaxf(acc[m][b][j] * sc, 0.f));
                __builtin_amdgcn_raw_buffer_store_b128(v, ro, ob + (unsigned)((m * 16 + j) * NN * 4), 0, OUT_AUX);
            }
        }
    }
}

__global__ void __launch_bounds__(512, 2) fused_kernel(Params P) {
    extern __shared__ __attribute__((aligned(16))) unsigned char smem[];
    LAS unsigned char* lds = (LAS unsigned char*)smem;
    STAMP(0);
    if (blockIdx.x < NPUSH) pusher(P, lds);
    else gemm(P, lds);
}

extern "C" void kernel_launch(int rank, int nranks, ncclComm_t comm,
                              hipStream_t stream,
                              void* const* d_in, const int* in_sizes, int n_in,
                              void* d_out, int out_size,
                              const int* in_dtypes, int out_dtype,
                              void* const* d_out_all,
                              void* const* d_ws_all, size_t ws_bytes) {
    if (nranks != NR || ws_bytes < WS_NEED || n_in != 4 || (unsigned)rank >= (unsigned)NR) {
        fprintf(stderr, "unsupported configuration\n");
        return;
    }
    static bool attr_set = false;
    if (!attr_set) {
        HIP_CHECK(hipFuncSetAttribute((const void*)fused_kernel, hipFuncAttributeMaxDynamicSharedMemorySize, LDS_BYTES));
        attr_set = true;
    }
    Params P;
    P.x = (const unsigned char*)d_in[0];
    P.w = (const unsigned char*)d_in[1];
    P.sx = (const float*)d_in[2];
    P.sw = (const float*)d_in[3];
    P.out = (float*)d_out;
    for (int d = 0; d < NR; ++d) P.ws[d] = (unsigned char*)d_ws_all[(rank + d) % NR];
    P.rank = rank;
    hipLaunchKernelGGL(fused_kernel, dim3(GRID), dim3(512), LDS_BYTES, stream, P);
    HIP_CHECK(hipGetLastError());
}
```

```cpp
#include <hip/hip_runtime.h>
#include <rccl/rccl.h>
#include <cstdio>
#include <cstdint>

#define HIP_CHECK(expr) do { hipError_t _e = (expr); if (_e != hipSuccess) { \
    fprintf(stderr, "HIP error %s at %s:%d\n", hipGetErrorString(_e), __FILE__, __LINE__); \
    return; } } while (0)

typedef int      v8i __attribute__((ext_vector_type(8)));
typedef int      v4i __attribute__((ext_vector_type(4)));
typedef float    v4f __attribute__((ext_vector_type(4)));
typedef unsigned v4u __attribute__((ext_vector_type(4)));
#define LAS __attribute__((address_space(3)))

constexpr int NR    = 8;
constexpr int MM    = 4096;
constexpr int NN    = 4096;
constexpr int KS    = 512;
constexpr int NSTEP = 32;
constexpr int NPUSH = 16;
constexpr int NRB   = 15;
constexpr int NCB   = 16;
constexpr int NGEMM = NRB * NCB;
constexpr int GRID  = NPUSH + NGEMM;

constexpr size_t WS_TMO   = 0;
constexpr size_t WS_DONE  = 512;
constexpr size_t WS_TICKET = 1024;
constexpr size_t WS_CTR   = 16 * 1024;
constexpr size_t WS_FLAGS = 64 * 1024;
constexpr size_t WS_PROF  = 128 * 1024;
constexpr size_t WS_DATA  = 1u << 20;
constexpr size_t PANEL    = 4096 * 16;
constexpr size_t OPER     = (size_t)NSTEP * NR * PANEL;
constexpr size_t BUFB     = 2 * OPER;
constexpr size_t WS_NEED  = WS_DATA + 2 * BUFB;

constexpr int A_PAN   = 288 * 16;
constexpr int B_PAN   = 256 * 16;
constexpr int A_STG   = NR * A_PAN;
constexpr int B_STG   = NR * B_PAN;
constexpr int STG     = A_STG + B_STG;
constexpr int LDS_MISC = 2 * STG;
constexpr int LDS_BYTES = LDS_MISC + 512;

#define OUT_AUX 16
constexpr int NL    = 2;
constexpr int NFULL = NSTEP - NL;
constexpr unsigned long long DEADLINE = 40000000ull;

struct Params {
    const unsigned char* x;
    const unsigned char* w;
    const float* sx;
    const float* sw;
    float* out;
    unsigned char* ws[NR];
    int rank;
};

__device__ __forceinline__ void ld_flag_async(unsigned& f, const unsigned* p) {
    asm volatile("global_load_dword %0, %1, off sc0 sc1" : "+v"(f) : "v"(p) : "memory");
}
__device__ __forceinline__ void ld_flag_async_masked(unsigned& f, const unsigned* p, int mask  ) {
    unsigned long long ex;
    asm volatile("s_mov_b64 %1, exec\n\ts_mov_b32 exec_lo, %3\n\ts_mov_b32 exec_hi, %3\n\t"
                 "global_load_dword %0, %2, off sc0 sc1\n\ts_mov_b64 exec, %1"
                 : "+v"(f), "=&s"(ex) : "v"(p), "s"(mask) : "memory");
}
__device__ __forceinline__ unsigned ld_flag(const unsigned* p) {
    unsigned f;
    asm volatile("global_load_dword %0, %1, off sc0 sc1\n\ts_waitcnt vmcnt(0)" : "=&v"(f) : "v"(p) : "memory");
    return f;
}

__device__ __forceinline__ int uni32(int v) {
    int r;
    asm volatile("s_nop 0\n\tv_readfirstlane_b32 %0, %1\n\ts_nop 4" : "=s"(r) : "v"(v));
    return r;
}
__device__ __forceinline__ unsigned long long uni64(unsigned long long v) {
    const unsigned lo = (unsigned)uni32((int)(unsigned)v), hi = (unsigned)uni32((int)(unsigned)(v >> 32));
    return ((unsigned long long)hi << 32) | lo;
}

#define STAMP(i) do {} while (0)

__device__ __forceinline__ unsigned next_call(const Params& P) {
    unsigned* c = (unsigned*)(P.ws[0] + WS_CTR + (size_t)blockIdx.x * 64);
    return __hip_atomic_load(c, __ATOMIC_RELAXED, __HIP_MEMORY_SCOPE_AGENT) + 1u;
}
__device__ __forceinline__ void commit_call(const Params& P, unsigned call) {
    unsigned* c = (unsigned*)(P.ws[0] + WS_CTR + (size_t)blockIdx.x * 64);
    __hip_atomic_store(c, call, __ATOMIC_RELAXED, __HIP_MEMORY_SCOPE_AGENT);
}

__device__ __forceinline__ v4u w_transpose(unsigned w0, unsigned w1, unsigned w2, unsigned w3, int b) {
    const unsigned l01 = __builtin_amdgcn_perm(w1, w0, 0x05010400u), h01 = __builtin_amdgcn_perm(w1, w0, 0x07030602u);
    const unsigned l23 = __builtin_amdgcn_perm(w3, w2, 0x05010400u), h23 = __builtin_amdgcn_perm(w3, w2, 0x07030602u);
    const unsigned o0 = __builtin_amdgcn_perm(l23, l01, 0x05040100u), o1 = __builtin_amdgcn_perm(l23, l01, 0x07060302u);
    const unsigned o2 = __builtin_amdgcn_perm(h23, h01, 0x05040100u), o3 = __builtin_amdgcn_perm(h23, h01, 0x07060302u);
#define QB(x, j) ((unsigned)__builtin_amdgcn_update_dpp(0, (int)(x), (j) * 0x55, 0xf, 0xf, true))
    const unsigned t00 = QB(o0, 0), t01 = QB(o1, 0), t02 = QB(o2, 0), t03 = QB(o3, 0);
    const unsigned t10 = QB(o0, 1), t11 = QB(o1, 1), t12 = QB(o2, 1), t13 = QB(o3, 1);
    const unsigned t20 = QB(o0, 2), t21 = QB(o1, 2), t22 = QB(o2, 2), t23 = QB(o3, 2);
    const unsigned t30 = QB(o0, 3), t31 = QB(o1, 3), t32 = QB(o2, 3), t33 = QB(o3, 3);
    v4u r;
    r.x = b == 0 ? t00 : b == 1 ? t01 : b == 2 ? t02 : t03;
    r.y = b == 0 ? t10 : b == 1 ? t11 : b == 2 ? t12 : t13;
    r.z = b == 0 ? t20 : b == 1 ? t21 : b == 2 ? t22 : t23;
    r.w = b == 0 ? t30 : b == 1 ? t31 : b == 2 ? t32 : t33;
    return r;
}
__device__ __forceinline__ unsigned w_src_off(int tid) {
    return (unsigned)((tid & 3) * 4 * NN + (tid >> 5) * 64 + ((tid >> 2) & 7) * 4);
}
__device__ __forceinline__ unsigned w_pos(int tid) {
    const int gp = tid >> 5, r = (tid >> 2) & 7, b = tid & 3;
    return (unsigned)((gp >> 2) * 2048 + ((b & 1) * 64 + (gp & 3) * 16 + 2 * r + (b >> 1)) * 16);
}

constexpr int NRELAY = 128;
__device__ __forceinline__ void relayout(const Params& P, int g, unsigned call) {
    const int tid = threadIdx.x;
    const int isw = g & 1, h = g >> 1;
    const __amdgpu_buffer_rsrc_t r0 = __builtin_amdgcn_make_buffer_rsrc(P.ws[0], 0, (int)WS_NEED, 0x00020000);
    const unsigned base = (unsigned)(WS_DATA + (size_t)(call & 1) * BUFB + (size_t)P.rank * PANEL);
    int fsl, fidx;
    if (!isw) {
        const int j = h & 7, rho = h >> 3;
        v4u v[4];
#pragma unroll
        for (int p = 0; p < 4; ++p)
            v[p] = *(const v4u*)(P.x + (size_t)(j * 512 + p * 128 + (tid >> 2)) * KS + rho * 64 + (tid & 3) * 16);
#pragma unroll
        for (int p = 0; p < 4; ++p)
            __builtin_amdgcn_raw_buffer_store_b128(v[p], r0, base + (unsigned)(rho * 4 + (tid & 3)) * (unsigned)(NR * PANEL)
                                                   + (unsigned)(j * 512 + p * 128 + (tid >> 2)) * 16u, 0, 17);
        fsl = rho * 4 + tid; fidx = j;
    } else {
        const int q = h & 3, r2 = h >> 2;
        unsigned d[4][4];
#pragma unroll
        for (int p = 0; p < 4; ++p)
#pragma unroll
            for (int k = 0; k < 4; ++k)
                d[p][k] = *(const unsigned*)(P.w + (size_t)((r2 * 2 + (p >> 1)) * 16 + k) * NN + q * 1024 + (p & 1) * 32 + w_src_off(tid));
#pragma unroll
        for (int p = 0; p < 4; ++p)
            __builtin_amdgcn_raw_buffer_store_b128(w_transpose(d[p][0], d[p][1], d[p][2], d[p][3], tid & 3), r0,
                                                   base + (unsigned)OPER + (unsigned)(r2 * 2 + (p >> 1)) * (unsigned)(NR * PANEL)
                                                   + (unsigned)(((p & 1) * 4 + q) * 8192) + w_pos(tid), 0, 17);
        fsl = r2 * 2 + (tid >> 1); fidx = 8 + (tid & 1) * 4 + q;
    }
    asm volatile("s_waitcnt vmcnt(0)" ::: "memory");
    __builtin_amdgcn_s_barrier();
    asm volatile("" ::: "memory");
    if (tid < 4)
        __hip_atomic_store((unsigned*)(P.ws[0] + WS_FLAGS + (size_t)fsl * (NR * 128) + (size_t)P.rank * 128 + fidx * 4),
                           call, __ATOMIC_RELAXED, __HIP_MEMORY_SCOPE_SYSTEM);
    if (tid == 64)
        __hip_atomic_fetch_add((unsigned*)(P.ws[0] + WS_DONE), 1u, __ATOMIC_RELAXED, __HIP_MEMORY_SCOPE_AGENT);
}

struct Raw { v4u x; unsigned w0, w1, w2, w3; };
constexpr int NDIRECT = 64;
constexpr int NUNIT = NSTEP * NPUSH;

__device__ __forceinline__ int unit_code(int p) {
    if (p < 16 * NFULL) return p;
    const int q = p - 16 * NFULL;
    if (q < 12 * NL) return (NFULL + q / 12) * 16 + q % 12;
    const int r = (q - 12 * NL) & 63;
    return (NFULL + (r >> 2)) * 16 + 12 + (r & 3);
}

__device__ __forceinline__ void pusher(const Params& P, LAS unsigned char* lds) {
    const int tid  = threadIdx.x;
    const int lane = tid & 63;
    const int wave = __builtin_amdgcn_readfirstlane(tid >> 6);
    const int pid = uni32((int)blockIdx.x);
    const int rank = P.rank;
    static_assert(NUNIT == 0x200 && NDIRECT == 64, "asm below");
    LAS unsigned* tk_lds = (LAS unsigned*)(lds + LDS_MISC);

    const unsigned xoff = (unsigned)tid * KS, poff = (unsigned)tid * 16u;
    const unsigned woffA = w_src_off(tid) + NN, woffB = woffA + 2 * NN;
    const unsigned long long xb = uni64((unsigned long long)P.x), wb = uni64((unsigned long long)P.w);
    auto unit_off = [&](int u) -> unsigned {
        return (unsigned)(u >> 4) * (unsigned)(NR * PANEL) + (unsigned)(u & 7) * 8192u + ((u & 8) ? (unsigned)OPER : 0u);
    };
    auto load_unit = [&](Raw& r, int p  , int u  , unsigned long long pbase) {
        const int s = u >> 4, j = u & 7;
        const unsigned long long xa = xb + (unsigned long long)(j * 512 * KS + s * 16);
        const unsigned long long wa = wb + (unsigned long long)(s * 16 * NN + (j & 3) * 1024 + (j >> 2) * 32);
        const unsigned long long pa = pbase + unit_off(u);
        asm volatile("s_cmpk_lt_i32 %12, 0x200\n\t"
                     "s_cbranch_scc0 3f\n\t"
                     "s_cmpk_lt_i32 %12, 64\n\t"
                     "s_cbranch_scc1 1f\n\t"
                     "global_load_dwordx4 %0, %8, %11 sc1\n\t"
                     "s_branch 3f\n"
                     "1:\n\t"
                     "s_bitcmp0_b32 %13, 3\n\t"
                     "s_cbranch_scc0 2f\n\t"
                     "global_load_dwordx4 %0, %5, %9\n\t"
                     "s_branch 3f\n"
                     "2:\n\t"
                     "global_load_dword %1, %6, %10 offset:-4096\n\t"
                     "global_load_dword %2, %6, %10\n\t"
                     "global_load_dword %3, %7, %10 offset:-4096\n\t"
                     "global_load_dword %4, %7, %10\n"
                     "3:"
                     : "+v"(r.x), "+v"(r.w0), "+v"(r.w1), "+v"(r.w2), "+v"(r.w3)
                     : "v"(xoff), "v"(woffA), "v"(woffB), "v"(poff), "s"(xa), "s"(wa), "s"(pa), "s"(p), "s"(u)
                     : "memory", "scc");
    };

    Raw ra = {{0u, 0u, 0u, 0u}, 0u, 0u, 0u, 0u}, rb = ra;
    int p0 = pid, p1 = NPUSH + pid, p2 = 2 * NPUSH + pid, p3 = 3 * NPUSH + pid;
    int c0 = p0, c1 = p1, c2 = p2, cp = -1;
    load_unit(ra, p0, c0, xb);
    load_unit(rb, p1, c1, xb);
    const unsigned call = __builtin_amdgcn_readfirstlane(next_call(P));
    unsigned* tkc = (unsigned*)(P.ws[0] + WS_TICKET + (size_t)(call & 1) * 128);
    if (pid == 0 && tid == 0)
        __hip_atomic_store((unsigned*)(P.ws[0] + WS_TICKET + (size_t)((call + 1) & 1) * 128), 0u, __ATOMIC_RELAXED, __HIP_MEMORY_SCOPE_AGENT);

    __amdgpu_buffer_rsrc_t rs[NR];
#pragma unroll
    for (int dd = 1; dd < NR; ++dd) rs[dd] = __builtin_amdgcn_make_buffer_rsrc(P.ws[dd], 0, (int)WS_NEED, 0x00020000);
    unsigned char* fb = P.ws[1];
#pragma unroll
    for (int q = 2; q < NR; ++q) fb = ((lane & 7) == q) ? P.ws[q] : fb;
    fb += WS_FLAGS + (size_t)rank * 128;
    const unsigned dbase = (unsigned)(WS_DATA + (size_t)(call & 1) * BUFB + (size_t)rank * PANEL);
    const unsigned long long pbase = uni64((unsigned long long)(P.ws[0] + dbase));
    const unsigned wpos = w_pos(tid);
    auto publish = [&](int q) {
        if (wave == 1 + ((q >> 4) & 3) && lane >= 1 && lane < NR)
            __hip_atomic_store((unsigned*)(fb + (size_t)(q >> 4) * (NR * 128) + (q & 15) * 4), call, __ATOMIC_RELAXED, __HIP_MEMORY_SCOPE_SYSTEM);
    };

    unsigned tka = 0, tkb = 0;
    asm volatile("s_waitcnt vmcnt(0)" : "+v"(ra.x), "+v"(ra.w0), "+v"(ra.w1), "+v"(ra.w2), "+v"(ra.w3),
                                        "+v"(rb.x), "+v"(rb.w0), "+v"(rb.w1), "+v"(rb.w2), "+v"(rb.w3) :: "memory");
    int it = 0;
    auto step = [&](Raw& rc, Raw& ro, unsigned& tkn, unsigned& tko) {
        const bool dw = (p0 < NDIRECT) && (c0 & 8);
        v4u item = rc.x;
        if (dw) item = w_transpose(rc.w0, rc.w1, rc.w2, rc.w3, tid & 3);
        const unsigned off = dbase + unit_off(c0) + (dw ? wpos : poff);
#pragma unroll
        for (int dd = 1; dd < NR; ++dd)
            __builtin_amdgcn_raw_buffer_store_b128(item, rs[dd], off, 0, 17);
        if (it == 2) {
            const unsigned* dn = (const unsigned*)(P.ws[0] + WS_DONE);
            const unsigned long long t0 = __builtin_amdgcn_s_memrealtime();
            while (ld_flag(dn) - call * (unsigned)NRELAY > 0x7fffffffu) {
                if (__builtin_amdgcn_s_memrealtime() - t0 > DEADLINE) {
                    __hip_atomic_store((unsigned*)(P.ws[0] + WS_TMO), call, __ATOMIC_RELAXED, __HIP_MEMORY_SCOPE_AGENT);
                    break;
                }
                __builtin_amdgcn_s_sleep(1);
            }
        }
        load_unit(rc, p2, c2, pbase);
        if (wave == 0) {
            unsigned long long ex;
            asm volatile("s_mov_b64 %1, exec\n\ts_mov_b64 exec, 1\n\t"
                         "global_atomic_add %0, %2, %3, off sc0\n\ts_mov_b64 exec, %1"
                         : "+v"(tkn), "=&s"(ex) : "v"(tkc), "v"(1u) : "memory");
        }
        if (p2 >= NUNIT)                     asm volatile("s_waitcnt vmcnt(7)" ::: "memory");
        else if (p2 < NDIRECT && (c2 & 8))   asm volatile("s_waitcnt vmcnt(11)" ::: "memory");
        else                                 asm volatile("s_waitcnt vmcnt(8)" ::: "memory");
        asm volatile("" : "+v"(tko), "+v"(ro.x), "+v"(ro.w0), "+v"(ro.w1), "+v"(ro.w2), "+v"(ro.w3));
        if (tid == 0) tk_lds[it & 1] = tko;
        asm volatile("s_waitcnt lgkmcnt(0)" ::: "memory");
        __builtin_amdgcn_s_barrier();
        asm volatile("" ::: "memory");
        if (it == 0 && tid == 0) commit_call(P, call);
        if (cp >= 0) publish(cp);
        cp = c0; c0 = c1; c1 = c2; c2 = unit_code(p3 < NUNIT ? p3 : 0);
        p0 = p1; p1 = p2; p2 = p3;
        if (it == 0) {
            p3 = 4 * NPUSH + pid;
        } else {
            const unsigned t = __builtin_amdgcn_readfirstlane(tk_lds[it & 1]);
            p3 = 5 * NPUSH + (int)(t < 4096u ? t : 4096u);
        }
        ++it;
    };
    while (true) {
        step(ra, rb, tka, tkb);
        if (p0 >= NUNIT) break;
        step(rb, ra, tkb, tka);
        if (p0 >= NUNIT) break;
    }
    asm volatile("s_waitcnt vmcnt(0)" : "+v"(tka), "+v"(tkb) :: "memory");
    __builtin_amdgcn_s_barrier();
    asm volatile("" ::: "memory");
    publish(cp);
}

#define MFMA(a, b, c) __builtin_amdgcn_mfma_scale_f32_16x16x128_f8f6f4((a), (b), (c), 0, 0, 0, 0, 0, 0)

__device__ __forceinline__ v8i ld_frag(const LAS unsigned char* p, int pan) {
    const v4i lo = *(const LAS v4i*)(p);
    const v4i hi = *(const LAS v4i*)(p + pan);
    return __builtin_shufflevector(lo, hi, 0, 1, 2, 3, 4, 5, 6, 7);
}

template <int V> struct IC { static constexpr int value = V; };

__device__ __forceinline__ void gemm(const Params& P, LAS unsigned char* lds) {
    const unsigned call = __builtin_amdgcn_readfirstlane(next_call(P));
    const unsigned fcall = call;
    const int tid  = threadIdx.x;
    const int lane = tid & 63;
    const int wave = __builtin_amdgcn_readfirstlane(tid >> 6);
    const int wm = wave >> 2, wn = wave & 3;
    const int g   = blockIdx.x - NPUSH;
    if (g < NRELAY) relayout(P, g, call);
    const int xcd = g & 7, idx = g >> 3;
    const int cb = 2 * xcd + (idx & 1), rb = idx >> 1;
    const int row0 = rb == 0 ? 0 : 16 + rb * 272, col0 = cb * 256;
    const int nunits = (rb == 0) ? 18 : 17;
    const bool has9 = (wm == 0) || (nunits == 18);
    unsigned char* ws = P.ws[0];
    unsigned* tmo = (unsigned*)(ws + WS_TMO);

    const int jx0 = row0 >> 9, jx1 = (row0 + nunits * 16 - 1) >> 9, jwe = 8 + (cb >> 2), jwl = 12 + (cb >> 2);
    const unsigned char* fbase = ws + WS_FLAGS + wave * 128;
    const unsigned fo_full  = (unsigned)(lane == 0 ? jx0 : lane == 1 ? jx1 : lane == 2 ? jwe : jwl) * 4u;
    const unsigned fo_early = (unsigned)(lane == 0 ? jx0 : lane == 1 ? jx1 : jwe) * 4u;
    const unsigned fo_late  = (unsigned)jwl * 4u + (unsigned)(NFULL + (lane & 1)) * (NR * 128);
    auto flag_ptr = [&](int v) -> const unsigned* {
        const unsigned o = v < NSTEP ? (v < NFULL ? fo_full : fo_early) + (unsigned)v * (NR * 128) : fo_late;
        return (const unsigned*)(fbase + (size_t)o);
    };

    const unsigned long long ga = uni64((unsigned long long)(ws + WS_DATA + (size_t)(call & 1) * BUFB + (size_t)wave * PANEL + (size_t)row0 * 16 + 2048));
    const unsigned long long gb = uni64((unsigned long long)(ws + WS_DATA + (size_t)(call & 1) * BUFB + OPER + (size_t)wave * PANEL + (size_t)cb * 2048));
    const unsigned voff = lane * 16;
    const unsigned long long m5 = uni64((nunits == 18) ? 0xffffffffull : 0xffffull);
    const unsigned lbase = (unsigned)(size_t)lds;
    const unsigned la = (unsigned)uni32((int)(lbase + wave * A_PAN + 2048));
    const unsigned lb = (unsigned)uni32((int)(lbase + A_STG + wave * B_PAN));

    auto issue = [&](int v) {
        const int sl = v < NSTEP ? v : NFULL;
        const unsigned long long pa = ga + (unsigned long long)sl * (NR * PANEL);
        const unsigned long long pbe = gb + (unsigned long long)sl * (NR * PANEL);
        const unsigned long long pbl = pbe + (32768 - 2048);
        const unsigned long long pbl2 = pbl + NR * PANEL;
        const unsigned bo = (v & 1) * STG;
        unsigned keep; unsigned long long ex;
        asm volatile(
            "s_mov_b32 %0, m0\n\t"
            "s_mov_b32 m0, %6\n\t"
            "s_cmpk_lt_i32 %9, %10\n\t"
            "s_cbranch_scc1 1f\n\t"
            "global_load_lds_dwordx4 %2, %5 offset:2048\n\t"
            "global_load_lds_dwordx4 %2, %5 offset:3072\n\t"
            "s_mov_b32 m0, %13\n\t"
            "s_nop 0\n\t"
            "global_load_lds_dwordx4 %2, %12 offset:2048\n\t"
            "global_load_lds_dwordx4 %2, %12 offset:3072\n\t"
            "s_branch 3f\n"
            "1:\n\t"
            "global_load_lds_dwordx4 %2, %4\n\t"
            "global_load_lds_dwordx4 %2, %4 offset:1024\n\t"
            "s_cmpk_lt_i32 %9, %11\n\t"
            "s_cbranch_scc0 2f\n\t"
            "global_load_lds_dwordx4 %2, %5 offset:2048\n\t"
            "global_load_lds_dwordx4 %2, %5 offset:3072\n"
            "2:\n\t"
            "s_mov_b32 m0, %7\n\t"
            "s_mov_b64 %1, exec\n\t"
            "global_load_lds_dwordx4 %2, %3 offset:-2048\n\t"
            "global_load_lds_dwordx4 %2, %3 offset:-1024\n\t"
            "global_load_lds_dwordx4 %2, %3\n\t"
            "global_load_lds_dwordx4 %2, %3 offset:1024\n\t"
            "s_mov_b64 exec, %8\n\t"
            "global_load_lds_dwordx4 %2, %3 offset:2048\n\t"
            "s_mov_b64 exec, %1\n"
            "3:\n\t"
            "s_mov_b32 m0, %0"
            : "=&s"(keep), "=&s"(ex)
            : "v"(voff), "s"(pa), "s"(pbe), "s"(pbl), "s"(lb + bo), "s"(la + bo), "s"(m5), "s"(v), "i"(NSTEP), "i"(NFULL),
              "s"(pbl2), "s"(lb + STG)
            : "memory", "scc");
    };
    static_assert(NSTEP % 2 == 0 && NFULL % 2 == 0, "buffer parity");
    bool dead = false;
    auto wait_flags = [&](const unsigned* fp, unsigned f) {
        if (__all(f >= fcall) || dead) return;
        const unsigned long long t0 = __builtin_amdgcn_s_memrealtime();
        for (unsigned it = 1;; ++it) {
            __builtin_amdgcn_s_sleep(2);
            f = ld_flag(fp);
            if (__all(f >= fcall)) return;
            if ((it & 63u) == 0u) {
                if (__hip_atomic_load(tmo, __ATOMIC_RELAXED, __HIP_MEMORY_SCOPE_AGENT) == call
                    || __builtin_amdgcn_s_memrealtime() - t0 > DEADLINE) {
                    __hip_atomic_store(tmo, call, __ATOMIC_RELAXED, __HIP_MEMORY_SCOPE_AGENT);
                    dead = true;
                    return;
                }
            }
        }
    };

    v4f acc[9][4];
#pragma unroll
    for (int m = 0; m < 9; ++m)
#pragma unroll
        for (int b = 0; b < 4; ++b) acc[m][b] = v4f{0.f, 0.f, 0.f, 0.f};

    const int fr = lane & 15, fg = lane >> 4;
    const LAS unsigned char* pa[2];
    const LAS unsigned char* pb[2];
    pa[0] = lds + (2 * fg) * A_PAN + (wm * 144 + fr) * 16;
    pb[0] = lds + A_STG + (2 * fg) * B_PAN + (wn * 16 + fr) * 16;
    pa[1] = pa[0] + STG;
    pb[1] = pb[0] + STG;
    asm volatile("" : "+v"(pa[0]), "+v"(pb[0]), "+v"(pa[1]), "+v"(pb[1]));

    const float sc = P.sx[0] * P.sw[0];
    const __amdgpu_buffer_rsrc_t ro = __builtin_amdgcn_make_buffer_rsrc(P.out, 0, MM * NN * 4, 0x00020000);
    const unsigned ob = (unsigned)((row0 + wm * 144 + fg * 4 + (fr & 1)) * NN + col0 + wn * 64 + (fr >> 1) * 4) * 4u;
    bool bad = false;
    auto store_pair = [&](auto CLS_, auto T_) __attribute__((always_inline)) {
        constexpr int cls = decltype(CLS_)::value, m = decltype(T_)::value >> 1, j0 = (decltype(T_)::value & 1) * 2;
        if (m < 8 || has9) {
            const bool odd = fr & 1;
            const float e0 = fmaxf(acc[m][2 * cls][j0] * sc, 0.f),     e1 = fmaxf(acc[m][2 * cls + 1][j0] * sc, 0.f);
            const float o0 = fmaxf(acc[m][2 * cls][j0 + 1] * sc, 0.f), o1 = fmaxf(acc[m][2 * cls + 1][j0 + 1] * sc, 0.f);
            const int s0 = __float_as_int(odd ? e0 : o0), s1 = __float_as_int(odd ? e1 : o1);
            const unsigned r0 = (unsigned)__builtin_amdgcn_update_dpp(0, s0, 0xb1, 0xf, 0xf, true);
            const unsigned r1 = (unsigned)__builtin_amdgcn_update_dpp(0, s1, 0xb1, 0xf, 0xf, true);
            v4u v;
            v.x = odd ? r0 : __float_as_uint(e0);
            v.y = odd ? r1 : __float_as_uint(e1);
            v.z = odd ? __float_as_uint(o0) : r0;
            v.w = odd ? __float_as_uint(o1) : r1;
            if (bad) v = v4u{0x7fc00000u, 0x7fc00000u, 0x7fc00000u, 0x7fc00000u};
            __builtin_amdgcn_raw_buffer_store_b128(v, ro, ob + (unsigned)(((m * 16 + j0) * NN + cls * 32) * 4), 0, OUT_AUX);
        }
    };

    wait_flags(flag_ptr(0), ld_flag(flag_ptr(0)));
    issue(0);
    wait_flags(flag_ptr(1), ld_flag(flag_ptr(1)));
    issue(1);
    asm volatile("s_waitcnt vmcnt(9)" ::: "memory");
    __builtin_amdgcn_s_barrier();
    asm volatile("" ::: "memory");
    if (tid == 0) commit_call(P, call);

    v8i B[4], A[3];
    unsigned f = 0;
#pragma unroll
    for (int b = 0; b < 4; ++b) B[b] = ld_frag(pb[0] + b * 1024, B_PAN);
    A[0] = ld_frag(pa[0], A_PAN);
    A[1] = ld_frag(pa[0] + 256, A_PAN);

    int nxt = 2;
    int fs = -1;

    auto do_step = [&](auto TYPE_, auto H_, auto HN_, int v) __attribute__((always_inline)) {
        constexpr int TYPE = decltype(TYPE_)::value, h = decltype(H_)::value, hn = decltype(HN_)::value;
        constexpr int f1 = TYPE == 1 ? 2 : 4;
#pragma unroll
        for (int m = 0; m < 7; ++m) {
            if (m + 2 < 8 || has9) A[(m + 2) % 3] = ld_frag(pa[h] + (m + 2) * 256, A_PAN);
            if (m == 3) {
                if (nxt == v + 1) asm volatile("s_waitcnt vmcnt(0)" ::: "memory");
                asm volatile("" : "+v"(f));
                if (nxt == v + 1 && __all(f >= fcall)) { issue(v + 1); nxt = v + 2; }
                fs = nxt <= NSTEP ? nxt : NSTEP;
                ld_flag_async(f, flag_ptr(fs));
            }
#pragma unroll
            for (int b = 0; b < f1; ++b) acc[m][b] = MFMA(A[m % 3], B[b], acc[m][b]);
        }
        __builtin_amdgcn_sched_barrier(0);
        if (nxt == v + 1) asm volatile("s_waitcnt vmcnt(0)" ::: "memory");
        asm volatile("" : "+v"(f));
        if (nxt == v + 1) {
            wait_flags(flag_ptr(v + 1), f);
            issue(v + 1);
            nxt = v + 2;
        }
        asm volatile("s_waitcnt vmcnt(0) lgkmcnt(0)" : "+v"(f) :: "memory");
        __builtin_amdgcn_s_barrier();
        asm volatile("" ::: "memory");
        __builtin_amdgcn_sched_barrier(0);
        STAMP(2 + v);
        {
            const bool open = (nxt == v + 2) && (nxt <= NSTEP);
            const bool ok = open && (fs == nxt) && __all(f >= fcall);
            ld_flag_async_masked(f, flag_ptr(open ? nxt : 0), __builtin_amdgcn_readfirstlane((open && !ok) ? -1 : 0));
            if (ok) { issue(v + 2); nxt = v + 3; }
        }
        const v8i An = ld_frag(pa[hn], A_PAN);
        if (has9) {
#pragma unroll
            for (int b = 0; b < f1; ++b) acc[8][b] = MFMA(A[2], B[b], acc[8][b]);
        }
#pragma unroll
        for (int b = 0; b < 4; ++b) {
            if (b < f1) acc[7][b] = MFMA(A[1], B[b], acc[7][b]);
            B[b] = ld_frag(pb[hn] + b * 1024, B_PAN);
        }
        A[0] = An;
        A[1] = ld_frag(pa[hn] + 256, A_PAN);
    };
    auto late_step = [&](auto H_, auto LI_) __attribute__((always_inline)) {
        constexpr int h = decltype(H_)::value, T0 = decltype(LI_)::value * 9;
        constexpr bool last = decltype(LI_)::value == 1;
#define LM(m) { if (m + 2 < 8 || has9) A[(m + 2) % 3] = ld_frag(pa[h] + (m + 2) * 256, A_PAN); \
                acc[m][2] = MFMA(A[m % 3], B[2], acc[m][2]); acc[m][3] = MFMA(A[m % 3], B[3], acc[m][3]); \
                store_pair(IC<0>{}, IC<T0 + m>{}); }
        LM(0) LM(1) LM(2) LM(3) LM(4) LM(5) LM(6)
#undef LM
        v8i An;
        if (!last) An = ld_frag(pa[h ^ 1], A_PAN);
        if (has9) {
            acc[8][2] = MFMA(A[2], B[2], acc[8][2]);
            acc[8][3] = MFMA(A[2], B[3], acc[8][3]);
        }
        store_pair(IC<0>{}, IC<T0 + 7>{});
#pragma unroll
        for (int b = 2; b < 4; ++b) {
            acc[7][b] = MFMA(A[1], B[b], acc[7][b]);
            if (!last) B[b] = ld_frag(pb[h ^ 1] + b * 1024, B_PAN);
        }
        store_pair(IC<0>{}, IC<T0 + 8>{});
        if (!last) {
            A[0] = An;
            A[1] = ld_frag(pa[h ^ 1] + 256, A_PAN);
        }
    };

#pragma nounroll
    for (int v = 0; v < NFULL; v += 2) {
        do_step(IC<0>{}, IC<0>{}, IC<1>{}, v);
        do_step(IC<0>{}, IC<1>{}, IC<0>{}, v + 1);
    }
    do_step(IC<1>{}, IC<0>{}, IC<1>{}, NFULL);
    do_step(IC<1>{}, IC<1>{}, IC<1>{}, NFULL + 1);
    bad = __builtin_amdgcn_readfirstlane(__hip_atomic_load(tmo, __ATOMIC_RELAXED, __HIP_MEMORY_SCOPE_AGENT) == call);
    late_step(IC<1>{}, IC<0>{});
    late_step(IC<0>{}, IC<1>{});

    STAMP(60);
#define ST2(t) store_pair(IC<1>{}, IC<t>{}); store_pair(IC<1>{}, IC<t + 1>{});
    ST2(0) ST2(2) ST2(4) ST2(6) ST2(8) ST2(10) ST2(12) ST2(14) ST2(16)
#undef ST2
}

__global__ void __launch_bounds__(512, 2) fused_kernel(Params P) {
    extern __shared__ __attribute__((aligned(16))) unsigned char smem[];
    LAS unsigned char* lds = (LAS unsigned char*)smem;
    STAMP(0);
    if (blockIdx.x < NPUSH) pusher(P, lds);
    else gemm(P, lds);
}

extern "C" void kernel_launch(int rank, int nranks, ncclComm_t comm,
                              hipStream_t stream,
                              void* const* d_in, const int* in_sizes, int n_in,
                              void* d_out, int out_size,
                              const int* in_dtypes, int out_dtype,
                              void* const* d_out_all,
                              void* const* d_ws_all, size_t ws_bytes) {
    if (nranks != NR || ws_bytes < WS_NEED || n_in != 4 || (unsigned)rank >= (unsigned)NR) {
        fprintf(stderr, "unsupported configuration\n");
        return;
    }
    static bool attr_set = false;
    if (!attr_set) {
        HIP_CHECK(hipFuncSetAttribute((const void*)fused_kernel, hipFuncAttributeMaxDynamicSharedMemorySize, LDS_BYTES));
        attr_set = true;
    }
    Params P;
    P.x = (const unsigned char*)d_in[0];
    P.w = (const unsigned char*)d_in[1];
    P.sx = (const float*)d_in[2];
    P.sw = (const float*)d_in[3];
    P.out = (float*)d_out;
    for (int d = 0; d < NR; ++d) P.ws[d] = (unsigned char*)d_ws_all[(rank + d) % NR];
    P.rank = rank;
    hipLaunchKernelGGL(fused_kernel, dim3(GRID), dim3(512), LDS_BYTES, stream, P);
    HIP_CHECK(hipGetLastError());
}
```
